```python
import numpy as np

kernel_name = "gpu_ag_gemm_m4096_k4096_n16384_f32_gelu"


ABSMAX_RTOL = 0.001

COMPUTE_DTYPE = "float32"
EPILOGUE = "gelu"
IN_QUANT = None
OUT_QUANT = None
IN_DTYPES = {"x": "float32", "w_mat": "float32"}
WORLD_SIZE = 8

def setup_inputs():

    x = np.random.randn(4096, 4096).astype(np.float32)
    w_mat = (np.random.randn(4096, 16384) / np.sqrt(4096)).astype(np.float32)
    return {"x": x, "w_mat": w_mat}

def _epilogue(y):
    c = 0.7978845608028654
    return (0.5 * y * (1.0 + np.tanh(c * (y + 0.044715 * y ** 3)))).astype(np.float32)

def reference(x, w_mat):
    y = (x @ w_mat).astype(np.float32)
    return _epilogue(y)
```

```cpp
#include <hip/hip_runtime.h>
#include <rccl/rccl.h>
#include <cstdio>
#include <cstdint>
#include <type_traits>
#include <utility>

#define HIP_CHECK(expr) do { hipError_t _e = (expr); if (_e != hipSuccess) { \
    fprintf(stderr, "HIP error %s at %s:%d\n", hipGetErrorString(_e), __FILE__, __LINE__); \
    return; } } while (0)

typedef _Float16 h8 __attribute__((ext_vector_type(8)));
typedef float f4 __attribute__((ext_vector_type(4)));
typedef __attribute__((address_space(3))) char lds_char;
typedef __attribute__((address_space(3))) unsigned lds_u32;
typedef __attribute__((address_space(3))) h8 lds_h8;
typedef __attribute__((address_space(1))) unsigned g_u32;
typedef const __attribute__((address_space(1))) void* g_cptr;
typedef __attribute__((address_space(3))) void* l_ptr;

namespace cfg {
constexpr int NRANK = 8;
constexpr int M = 4096, K = 4096, N = 2048;
constexpr int MS = M / NRANK;
constexpr int BM = 256, BK = 64;
constexpr int MT = M / BM;
constexpr int NF = N / 16;
constexpr int NKS = K / BK;
constexpr int SPC = 4;
constexpr int NCH = NKS / SPC;
constexpr int NWG = 256;
#define GPX_ 30
constexpr int GPX = GPX_;
constexpr int NGW = 8 * GPX;
constexpr int GPR = GPX / 2;
constexpr int FLO = NF / GPR;
constexpr int NWIDE = NF - FLO * GPR;
constexpr int SPX = 32 - GPX;
constexpr int NSWG = NWG - NGW;
constexpr int SROWS = MS / NSWG;
#define NSW_ 2
constexpr int NSW = NSW_;
constexpr int SRING = 4;
constexpr int A_BYTES = BM * BK * 2;
constexpr int FR_BYTES = 16 * BK * 2;
constexpr int STAGE = A_BYTES + (FLO + 1) * FR_BYTES;
constexpr int NSTAGE = 3;
constexpr int CTL_OFF = NSTAGE * STAGE;
constexpr int FLG_OFF = CTL_OFF;
constexpr int DBG_OFF = CTL_OFF + 256;
constexpr int LDS_BYTES = DBG_OFF + 256;
constexpr size_t OFF_EPOCH = 0;
constexpr size_t OFF_DONE = 128;
constexpr size_t OFF_TMO = 256;
constexpr size_t OFF_LCNT = 1024;
constexpr size_t OFF_XFLAG = 65536;
constexpr size_t XFLAG_SRC = 64 * 32 * 4;
constexpr size_t OFF_W16 = 1u << 20;
constexpr size_t W16_BYTES = (size_t)K * N * 2;
constexpr size_t OFF_X16 = OFF_W16 + W16_BYTES;
constexpr size_t X16_BYTES = (size_t)M * K * 2;
constexpr size_t WS_NEED = OFF_X16 + 2 * X16_BYTES;
constexpr unsigned long long DEADLINE = 50000000ull;
#define DMA_AUX 16
}
using namespace cfg;

#define STAMP(slot) do { } while (0)
#define LSTAMP(slot) do { } while (0)
#define LDUMP(first) do { } while (0)

__device__ __forceinline__ float gelu_tanh(float y) {
    const float c2 = -2.0f * 0.7978845608028654f * 1.4426950408889634f;
    float t = y * y;
    float z = y * __builtin_fmaf(t, 0.044715f * c2, c2);
    float e = __builtin_amdgcn_exp2f(z);
    return y * __builtin_amdgcn_rcpf(1.0f + e);
}

#define BAR() __builtin_amdgcn_s_barrier()

template <int NFR>
__device__ __forceinline__ void gemm_tile(lds_char* smem, char* ws, float* __restrict__ out, int mt, int f0,
                                          int wid, int lane, unsigned call, int rank, int bid, bool& bad)
{
    constexpr int NDMA = 6 + (NFR - 8);
    const size_t x16_off = OFF_X16 + (size_t)(call & 1) * X16_BYTES;
    const char* ga = ws + x16_off + (size_t)mt * NKS * A_BYTES + wid * 4096 + lane * 16;
    const char* gb = ws + OFF_W16 + (size_t)f0 * FR_BYTES + wid * 2048 + lane * 16;
    const char* gx = ws + OFF_W16 + (size_t)f0 * FR_BYTES + 16384 + wid * 256 + lane * 4;
    const int fo = (lane & 15) * 128 + (((lane >> 4) ^ (lane & 7)) * 16);
    const int aoff = wid * 32 * 128 + fo;
    const int boff = A_BYTES + fo;
    g_u32* tmo = (g_u32*)(ws + OFF_TMO);
    const int src = mt >> 1;
    const bool remote = src != rank;
    constexpr int FPT = BM / SROWS;
    const char* fl = ws + OFF_XFLAG + (size_t)src * XFLAG_SRC + ((mt & 1) * FPT + (lane & (FPT - 1))) * 4;
    const unsigned long long t0 = __builtin_amdgcn_s_memrealtime();

    f4 acc[2][NFR];
    #pragma unroll
    for (int i = 0; i < 2; ++i)
        #pragma unroll
        for (int j = 0; j < NFR; ++j) acc[i][j] = (f4){0.f, 0.f, 0.f, 0.f};
    h8 a0[2], b0[NFR], a1[2], b1[NFR];

    auto issue = [&](int t, int st) {
        lds_char* sb = smem + st * STAGE;
        const char* pa = ga + (size_t)t * A_BYTES;
        const char* pb = gb + (size_t)t * (NF * FR_BYTES);
        lds_char* la = sb + wid * 4096;
        lds_char* lb = sb + A_BYTES + wid * 2048;
        __builtin_amdgcn_global_load_lds((g_cptr)pa, (l_ptr)la, 16, 0, DMA_AUX);
        __builtin_amdgcn_global_load_lds((g_cptr)pa, (l_ptr)la, 16, 1024, DMA_AUX);
        __builtin_amdgcn_global_load_lds((g_cptr)pa, (l_ptr)la, 16, 2048, DMA_AUX);
        __builtin_amdgcn_global_load_lds((g_cptr)pa, (l_ptr)la, 16, 3072, DMA_AUX);
        __builtin_amdgcn_global_load_lds((g_cptr)pb, (l_ptr)lb, 16, 0, DMA_AUX);
        __builtin_amdgcn_global_load_lds((g_cptr)pb, (l_ptr)lb, 16, 1024, DMA_AUX);
        if constexpr (NFR >= 9) {
            const char* px = gx + (size_t)t * (NF * FR_BYTES);
            __builtin_amdgcn_global_load_lds((g_cptr)px, (l_ptr)(sb + A_BYTES + 16384 + wid * 256), 4, 0, DMA_AUX);
            if constexpr (NFR >= 10)
                __builtin_amdgcn_global_load_lds((g_cptr)px, (l_ptr)(sb + A_BYTES + 16384 + wid * 256), 4, 2048, DMA_AUX);
        }
    };
    auto rd = [&](h8 (&a)[2], h8 (&b)[NFR], int st, int kh) {
        lds_char* sb = smem + st * STAGE;
        lds_char* pa = sb + (aoff ^ (kh * 64));
        lds_char* pb = sb + (boff ^ (kh * 64));
        a[0] = *(lds_h8*)(pa);
        b[0] = *(lds_h8*)(pb);
        a[1] = *(lds_h8*)(pa + 2048);
        #pragma unroll
        for (int j = 1; j < NFR; ++j) b[j] = *(lds_h8*)(pb + j * 2048);
    };
    auto mma = [&](h8 (&a)[2], h8 (&b)[NFR]) {
        #pragma unroll
        for (int j = 0; j < NFR; ++j)
            #pragma unroll
            for (int i = 0; i < 2; ++i)
                acc[i][j] = __builtin_amdgcn_mfma_f32_16x16x32_f16(b[j], a[i], acc[i][j], 0, 0, 0);
    };
    auto spin_flags = [&](int s) {
        const char* p = fl + (size_t)s * (NSWG * 4);
        for (unsigned it = 0;; ++it) {
            unsigned v;
            asm volatile("global_load_dword %0, %1, off sc0 sc1\n\ts_waitcnt vmcnt(0)" : "=v"(v) : "v"(p) : "memory");
            if (__all(v >= call)) break;
            if ((it & 15) == 15) {
                if (__hip_atomic_load(tmo, __ATOMIC_RELAXED, __HIP_MEMORY_SCOPE_AGENT) == call ||
                    __builtin_amdgcn_s_memrealtime() - t0 > DEADLINE) {
                    __hip_atomic_store(tmo, call, __ATOMIC_RELAXED, __HIP_MEMORY_SCOPE_AGENT);
                    break;
                }
            }
        }
    };
    auto pref_flags = [&](int s) {
        __builtin_amdgcn_global_load_lds((g_cptr)(fl + (size_t)s * (NSWG * 4)), (l_ptr)(smem + FLG_OFF), 4, 0, 17);
    };
    auto lds_flags_ok = [&]() {
        unsigned v;
        asm volatile("ds_read_b32 %0, %1\n\ts_waitcnt lgkmcnt(0)" : "=v"(v) : "v"((unsigned)(FLG_OFF + lane * 4)) : "memory");
        return __all(v >= call);
    };
    auto step = [&](int t, int st, auto arr_c, auto vm_c, auto iss_c) {
        constexpr bool ARR = decltype(arr_c)::value;
        constexpr int VM = decltype(vm_c)::value;
        constexpr bool ISS = decltype(iss_c)::value;
        const int st1 = st == 2 ? 0 : st + 1;
        rd(a1, b1, st, 1);
        mma(a0, b0);
        #pragma unroll
        for (int g = 0; g < NFR + 2; ++g) {
            __builtin_amdgcn_sched_group_barrier(0x008, 1, 0);
            __builtin_amdgcn_sched_group_barrier(0x100, 1, 0);
        }
        __builtin_amdgcn_sched_group_barrier(0x008, NFR - 2, 0);
        __builtin_amdgcn_sched_barrier(0);
        if constexpr (ARR) {
            asm volatile("s_waitcnt vmcnt(%0) lgkmcnt(0)" :: "i"(VM) : "memory");
            if constexpr (ISS) {
                if (remote && wid == 0) { if (!lds_flags_ok()) spin_flags(t + 3); }
            }
            BAR();
            __builtin_amdgcn_sched_barrier(0);
            if constexpr (ISS) {
                if (remote && wid == 0 && t + 4 < NKS) pref_flags(t + 4);
                issue(t + 3, st);
            }
            rd(a0, b0, st1, 0);
        }
        mma(a1, b1);
        if constexpr (ARR) {
            #pragma unroll
            for (int g = 0; g < NFR + 2; ++g) {
                __builtin_amdgcn_sched_group_barrier(0x008, 1, 1);
                if constexpr (ISS) { if (g < NDMA) __builtin_amdgcn_sched_group_barrier(0x020, 1, 1); }
                __builtin_amdgcn_sched_group_barrier(0x100, 1, 1);
            }
            __builtin_amdgcn_sched_group_barrier(0x008, NFR - 2, 1);
        }
        __builtin_amdgcn_sched_barrier(0);
    };
    using T_ = std::integral_constant<bool, true>;
    using F_ = std::integral_constant<bool, false>;
    using IN = std::integral_constant<int, NDMA>;
    using I0 = std::integral_constant<int, 0>;

    if (remote && wid == 0) { spin_flags(0); spin_flags(1); spin_flags(2); }
    BAR();
    if (wid == 0) STAMP(2);
    if (remote && wid == 0) pref_flags(3);
    issue(0, 0);
    issue(1, 1);
    issue(2, 2);
    asm volatile("s_waitcnt vmcnt(%0)" :: "i"(2 * NDMA) : "memory");
    BAR();
    __builtin_amdgcn_sched_barrier(0);
    rd(a0, b0, 0, 0);
    int st = 0;
    for (int t = 0; t < NKS - 3; ++t) {
        step(t, st, T_{}, IN{}, T_{});
        st = st == 2 ? 0 : st + 1;
    }
    step(NKS - 3, st, T_{}, IN{}, F_{}); st = st == 2 ? 0 : st + 1;
    step(NKS - 2, st, T_{}, I0{}, F_{}); st = st == 2 ? 0 : st + 1;
    step(NKS - 1, st, F_{}, I0{}, F_{});
    if (wid == 0) STAMP(3);

    bad = bad || __hip_atomic_load(tmo, __ATOMIC_RELAXED, __HIP_MEMORY_SCOPE_AGENT) == call;
    const float qnan = __builtin_nanf("");
    #pragma unroll
    for (int i = 0; i < 2; ++i) {
        const int m = mt * BM + wid * 32 + i * 16 + (lane & 15);
        #pragma unroll
        for (int j = 0; j < NFR; ++j) {
            const int n = (f0 + j) * 16 + (lane >> 4) * 4;
            f4 o;
            #pragma unroll
            for (int r = 0; r < 4; ++r) o[r] = bad ? qnan : gelu_tanh(acc[i][j][r]);
            *(f4*)(out + (size_t)m * N + n) = o;
        }
    }
}

__global__ void __launch_bounds__(512, 2)
ag_gemm_kernel(const float* __restrict__ x, const float* __restrict__ w, float* __restrict__ out,
               void* const* __restrict__ ws_all, int rank)
{
    __shared__ __attribute__((aligned(16))) char smem_raw[LDS_BYTES];
    lds_char* smem = (lds_char*)smem_raw;

    const int tid = threadIdx.x, lane = tid & 63;
    const int wid = __builtin_amdgcn_readfirstlane(tid >> 6);
    const int bid = blockIdx.x;
    const int xcd = bid & 7, idx = bid >> 3;
    char* ws = (char*)ws_all[rank];
    const unsigned call = __builtin_amdgcn_readfirstlane(
        __hip_atomic_load((g_u32*)(ws + OFF_EPOCH), __ATOMIC_RELAXED, __HIP_MEMORY_SCOPE_AGENT)) + 1;
    const size_t x16_off = OFF_X16 + (size_t)(call & 1) * X16_BYTES;
    if (wid == 0) STAMP(0);

    if (idx < GPX) {
        const int gwv = (xcd * GPX + idx) * 8 + wid;
        constexpr int NWV = NGW * 8;
        {
            const int kg = lane >> 3, ng = lane & 7;
            f4 v[3][8], u[3][2];
            #pragma unroll
            for (int it = 0; it < 3; ++it) {
                const int b = gwv + it * NWV;
                if (b < 4096) {
                    const int ks = b >> 6, nb = b & 63;
                    const float* p = w + (size_t)(ks * 64 + kg * 8) * N + nb * 32 + ng * 4;
                    #pragma unroll
                    for (int j = 0; j < 8; ++j) v[it][j] = *(const f4*)(p + (size_t)j * N);
                    const int xks = b & 63, rg = b >> 6;
                    const float* px = x + (size_t)(rg * 8 + kg) * K + xks * 64 + (ng ^ kg) * 8;
                    u[it][0] = *(const f4*)px; u[it][1] = *(const f4*)(px + 4);
                }
            }
            #pragma unroll
            for (int it = 0; it < 3; ++it) {
                const int b = gwv + it * NWV;
                if (b < 4096) {
                    const int ks = b >> 6, nb = b & 63;
                    #pragma unroll
                    for (int i = 0; i < 4; ++i) {
                        h8 h;
                        #pragma unroll
                        for (int j = 0; j < 8; ++j) h[j] = (_Float16)v[it][j][i];
                        const int n = nb * 32 + ng * 4 + i;
                        char* dst = ws + OFF_W16 + (size_t)(ks * NF + (n >> 4)) * FR_BYTES + (n & 15) * 128 + ((kg ^ (n & 7)) * 16);
                        asm volatile("global_store_dwordx4 %0, %1, off sc1\n\ts_nop 1" :: "v"(dst), "v"(h) : "memory");
                    }
                    const int xks = b & 63, rg = b >> 6;
                    h8 h;
                    #pragma unroll
                    for (int e = 0; e < 4; ++e) { h[e] = (_Float16)u[it][0][e]; h[4 + e] = (_Float16)u[it][1][e]; }
                    const int row = rg * 8 + kg;
                    char* dst = ws + x16_off + (size_t)((rank * 2 + (row >> 8)) * NKS + xks) * A_BYTES + (row & 255) * 128 + ng * 16;
                    asm volatile("global_store_dwordx4 %0, %1, off sc1\n\ts_nop 1" :: "v"(dst), "v"(h) : "memory");
                }
            }
            asm volatile("s_waitcnt vmcnt(0)" ::: "memory");
            BAR();
            if (wid == 0) STAMP(1);
        }
        bool bad = false;
        if (wid == 0) {
            if (lane == 0)
                __hip_atomic_fetch_add((g_u32*)(ws + OFF_LCNT + xcd * 128), 1u, __ATOMIC_RELAXED, __HIP_MEMORY_SCOPE_AGENT);
            g_u32* tmo = (g_u32*)(ws + OFF_TMO);
            g_u32* lc = (g_u32*)(ws + OFF_LCNT + (lane & 7) * 128);
            const unsigned long long t0 = __builtin_amdgcn_s_memrealtime();
            while (!__all(__hip_atomic_load(lc, __ATOMIC_RELAXED, __HIP_MEMORY_SCOPE_AGENT) >= call * GPX)) {
                if (__hip_atomic_load(tmo, __ATOMIC_RELAXED, __HIP_MEMORY_SCOPE_AGENT) == call ||
                    __builtin_amdgcn_s_memrealtime() - t0 > DEADLINE) {
                    __hip_atomic_store(tmo, call, __ATOMIC_RELAXED, __HIP_MEMORY_SCOPE_AGENT);
                    break;
                }
                __builtin_amdgcn_s_sleep(1);
            }
        }
        const int mt = xcd * 2 + idx / GPR, g = idx % GPR;
        if (g < NWIDE) gemm_tile<FLO + 1>(smem, ws, out, mt, (FLO + 1) * g, wid, lane, call, rank, bid, bad);
        else           gemm_tile<FLO>(smem, ws, out, mt, (FLO + 1) * NWIDE + FLO * (g - NWIDE), wid, lane, call, rank, bid, bad);
    } else {
        const int swg = xcd * SPX + (idx - GPX);
        const int r0 = swg * SROWS;
        const unsigned A_LOADED = CTL_OFF, A_FREED = CTL_OFF + 64;
        if (tid < 16) *(lds_u32*)(smem + CTL_OFF + tid * 4 + (tid ? 60 : 0)) = 0;
        __syncthreads();
        if (wid == 7) {
            const char* gsrc = (const char*)x + (size_t)r0 * (K * 4) + lane * 16;
            for (int c = 0; c < NCH; ++c) {
                if (c >= SRING) {
                    for (;;) {
                        unsigned v;
                        asm volatile("ds_read_b32 %0, %1\n\ts_waitcnt lgkmcnt(0)" : "=v"(v) : "v"(A_FREED + (c % SRING) * 4) : "memory");
                        if (__builtin_amdgcn_readfirstlane(v) >= (unsigned)((c / SRING) * SPC)) break;
                        __builtin_amdgcn_s_sleep(1);
                    }
                }
                lds_char* slot = smem + (c % SRING) * (SROWS * 1024);
                #pragma unroll
                for (int r = 0; r < SROWS; ++r)
                    __builtin_amdgcn_global_load_lds((g_cptr)(gsrc + (size_t)r * (K * 4) + c * 1024), (l_ptr)(slot + r * 1024), 16, 0, 0);
                if (c > 0) {
                    asm volatile("s_waitcnt vmcnt(%0)" :: "i"(SROWS) : "memory");
                    asm volatile("ds_write_b32 %0, %1" :: "v"(A_LOADED), "v"((unsigned)c) : "memory");
                }
            }
            asm volatile("s_waitcnt vmcnt(0)" ::: "memory");
            asm volatile("ds_write_b32 %0, %1" :: "v"(A_LOADED), "v"((unsigned)NCH) : "memory");
        } else if (wid < NSW) {
            const char* d0 = (const char*)ws_all[(rank + 1) & 7];
            const char* d1 = (const char*)ws_all[(rank + 2) & 7];
            const char* d2 = (const char*)ws_all[(rank + 3) & 7];
            const char* d3 = (const char*)ws_all[(rank + 4) & 7];
            const char* d4 = (const char*)ws_all[(rank + 5) & 7];
            const char* d5 = (const char*)ws_all[(rank + 6) & 7];
            const char* d6 = (const char*)ws_all[(rank + 7) & 7];
            char* fdst = (char*)ws_all[(rank + 1 + (lane & 7)) & 7] + OFF_XFLAG + (size_t)rank * XFLAG_SRC + (wid * NSWG + swg) * 4;
            asm volatile("" : "+v"(fdst));
            const int rl = lane >> 3, q = lane & 7;
            const unsigned lrd = rl * 1024 + (q ^ rl) * 32;
            const int mtg = rank * 2 + (r0 >> 8);
            unsigned doff = (unsigned)(x16_off + ((size_t)mtg * NKS + wid) * A_BYTES + (r0 & 255) * 128 + lane * 16);
            constexpr int NRG = SROWS / 8;
            #define ST1(d) asm volatile("global_store_dwordx4 %0, %1, %2 offset:%3 sc0 sc1\n\ts_nop 1" :: "v"(doff), "v"(h), "s"(d), "i"(rg * 1024) : "memory")
            for (int ks = wid; ks < NKS; ks += NSW) {
                const int c = ks / SPC;
                for (;;) {
                    unsigned v;
                    asm volatile("ds_read_b32 %0, %1\n\ts_waitcnt lgkmcnt(0)" : "=v"(v) : "v"(A_LOADED) : "memory");
                    if (__builtin_amdgcn_readfirstlane(v) > (unsigned)c) break;
                    __builtin_amdgcn_s_sleep(1);
                }
                const unsigned sb = (c % SRING) * (SROWS * 1024) + (ks % SPC) * 256 + lrd;
                f4 v[NRG][2];
                #pragma unroll
                for (int rg = 0; rg < NRG; ++rg) {
                    asm volatile("ds_read_b128 %0, %1 offset:%2" : "=v"(v[rg][0]) : "v"(sb), "i"(rg * 8192) : "memory");
                    asm volatile("ds_read_b128 %0, %1 offset:%2" : "=v"(v[rg][1]) : "v"(sb), "i"(rg * 8192 + 16) : "memory");
                }
                #pragma unroll
                for (int rg = 0; rg < NRG; ++rg) asm volatile("s_waitcnt lgkmcnt(0)" : "+v"(v[rg][0]), "+v"(v[rg][1]) :: "memory");
                if (lane == 0) asm volatile("ds_add_u32 %0, %1" :: "v"(A_FREED + (c % SRING) * 4), "v"(1u) : "memory");
                #pragma unroll
                for (int rg = 0; rg < NRG; ++rg) {
                    h8 h;
                    #pragma unroll
                    for (int e = 0; e < 4; ++e) { h[e] = (_Float16)v[rg][0][e]; h[4 + e] = (_Float16)v[rg][1][e]; }
                    ST1(d0); ST1(d1); ST1(d2); ST1(d3); ST1(d4); ST1(d5); ST1(d6);
                }
                doff += NSW * A_BYTES;
                asm volatile("s_waitcnt vmcnt(0)" ::: "memory");
                if (lane < 7) asm volatile("global_store_dword %0, %1, off sc0 sc1" :: "v"(fdst), "v"(call) : "memory");
                fdst += NSW * NSWG * 4;
                if ((ks % SPC) == SPC - 1) LSTAMP(c);
            }
            #undef ST1
        }
    }
    if (wid == 0 && lane == 0) {
        unsigned old = __hip_atomic_fetch_add((g_u32*)(ws + OFF_DONE), 1u, __ATOMIC_RELAXED, __HIP_MEMORY_SCOPE_AGENT);
        if ((old + 1) % NWG == 0)
            __hip_atomic_store((g_u32*)(ws + OFF_EPOCH), (old + 1) / NWG, __ATOMIC_RELAXED, __HIP_MEMORY_SCOPE_AGENT);
    }
}

extern "C" void kernel_launch(int rank, int nranks, ncclComm_t comm,
                              hipStream_t stream,
                              void* const* d_in, const int* in_sizes, int n_in,
                              void* d_out, int out_size,
                              const int* in_dtypes, int out_dtype,
                              void* const* d_out_all,
                              void* const* d_ws_all, size_t ws_bytes) {
    if (nranks != NRANK || n_in < 2 || ws_bytes < WS_NEED || in_sizes[0] != MS * K ||
        in_sizes[1] != K * N || out_size != M * N) {
        fprintf(stderr, "kernel_launch: unexpected problem geometry\n");
        return;
    }
    hipLaunchKernelGGL(ag_gemm_kernel, dim3(NWG), dim3(512), 0, stream,
                       (const float*)d_in[0], (const float*)d_in[1], (float*)d_out,
                       d_ws_all, rank);
    HIP_CHECK(hipGetLastError());
}
```

```cpp
#include <hip/hip_runtime.h>
#include <rccl/rccl.h>
#include <cstdio>
#include <cstdint>
#include <type_traits>
#include <utility>

#define HIP_CHECK(expr) do { hipError_t _e = (expr); if (_e != hipSuccess) { \
    fprintf(stderr, "HIP error %s at %s:%d\n", hipGetErrorString(_e), __FILE__, __LINE__); \
    return; } } while (0)

typedef _Float16 h8 __attribute__((ext_vector_type(8)));
typedef float f4 __attribute__((ext_vector_type(4)));
typedef __attribute__((address_space(3))) char lds_char;
typedef __attribute__((address_space(3))) unsigned lds_u32;
typedef __attribute__((address_space(3))) h8 lds_h8;
typedef __attribute__((address_space(1))) unsigned g_u32;
typedef const __attribute__((address_space(1))) void* g_cptr;
typedef __attribute__((address_space(3))) void* l_ptr;

namespace cfg {
constexpr int NRANK = 8;
constexpr int M = 4096, K = 4096, N = 2048;
constexpr int MS = M / NRANK;
constexpr int BM = 256, BK = 64;
constexpr int MT = M / BM;
constexpr int NF = N / 16;
constexpr int NKS = K / BK;
constexpr int SPC = 4;
constexpr int NCH = NKS / SPC;
constexpr int NWG = 256;
constexpr int GPX = 30;
constexpr int NGW = 8 * GPX;
constexpr int GPR = GPX / 2;
constexpr int FLO = NF / GPR;
constexpr int NWIDE = NF - FLO * GPR;
constexpr int SPX = 32 - GPX;
constexpr int NSWG = NWG - NGW;
constexpr int SROWS = MS / NSWG;
constexpr int NSW = 2;
constexpr int SRING = 4;
constexpr int A_BYTES = BM * BK * 2;
constexpr int FR_BYTES = 16 * BK * 2;
constexpr int STAGE = (FLO + 1) * FR_BYTES;
constexpr int NSTAGE = 4;
constexpr int SROW_LDS = 1024 + 16;
constexpr int CTL_OFF = SRING * SROWS * SROW_LDS > NSTAGE * STAGE ? SRING * SROWS * SROW_LDS : NSTAGE * STAGE;
constexpr int FLG_OFF = CTL_OFF;
constexpr int DBG_OFF = CTL_OFF + 256;
constexpr int LDS_BYTES = DBG_OFF + 256;
constexpr size_t OFF_EPOCH = 0;
constexpr size_t OFF_DONE = 128;
constexpr size_t OFF_TMO = 256;
constexpr size_t OFF_LCNT = 1024;
constexpr size_t OFF_XFLAG = 65536;
constexpr size_t XFLAG_SRC = 64 * 32 * 4;
constexpr size_t OFF_W16 = 1u << 20;
constexpr size_t W16_BYTES = (size_t)K * N * 2;
constexpr size_t OFF_X16 = OFF_W16 + W16_BYTES;
constexpr size_t X16_BYTES = (size_t)M * K * 2;
constexpr size_t WS_NEED = OFF_X16 + 2 * X16_BYTES;
constexpr unsigned long long DEADLINE = 50000000ull;
constexpr int GDELAY = 2;
constexpr int DMA_AUX = 16;
}
using namespace cfg;

#define STAMP(slot) do { } while (0)
#define LSTAMP(slot) do { } while (0)
#define LDUMP(first) do { } while (0)

__device__ __forceinline__ float gelu_tanh(float y) {
    const float c2 = -2.0f * 0.7978845608028654f * 1.4426950408889634f;
    float t = y * y;
    float z = y * __builtin_fmaf(t, 0.044715f * c2, c2);
    float e = __builtin_amdgcn_exp2f(z);
    return y * __builtin_amdgcn_rcpf(1.0f + e);
}

#define BAR() __builtin_amdgcn_s_barrier()

template <int NFR>
__device__ __forceinline__ void gemm_tile(lds_char* smem, char* ws, float* __restrict__ out, int mt, int f0,
                                          int wid, int lane, unsigned call, int rank, int bid, bool& bad)
{
    constexpr int NBD = 2 + (NFR - 8);
    constexpr int PER = 4 + NBD;
    const size_t x16_off = OFF_X16 + (size_t)(call & 1) * X16_BYTES;
    const char* ga = ws + x16_off + (size_t)mt * NKS * A_BYTES + wid * 4096 + lane * 16;
    const char* gb = ws + OFF_W16 + (size_t)f0 * FR_BYTES + wid * 2048 + lane * 16;
    const char* gx = ws + OFF_W16 + (size_t)f0 * FR_BYTES + 16384 + wid * 256 + lane * 4;
    const int boff = (lane & 15) * 128 + (((lane >> 4) ^ (lane & 7)) * 16);
    g_u32* tmo = (g_u32*)(ws + OFF_TMO);
    const int src = mt >> 1;
    const bool remote = src != rank;
    constexpr int FPT = BM / SROWS;
    const char* fl = ws + OFF_XFLAG + (size_t)src * XFLAG_SRC + ((mt & 1) * FPT + (lane & (FPT - 1))) * 4;
    const unsigned long long t0 = __builtin_amdgcn_s_memrealtime();

    f4 acc[2][NFR];
    #pragma unroll
    for (int i = 0; i < 2; ++i)
        #pragma unroll
        for (int j = 0; j < NFR; ++j) acc[i][j] = (f4){0.f, 0.f, 0.f, 0.f};
    h8 ar[4][4];
    h8 b0[NFR], b1[NFR];

    auto issue = [&](int t, auto sc) {
        constexpr int S = decltype(sc)::value;
        const char* pa = ga + (size_t)t * A_BYTES;
        asm volatile("global_load_dwordx4 %0, %1, off sc1" : "=v"(ar[S][0]) : "v"(pa) : "memory");
        asm volatile("global_load_dwordx4 %0, %1, off offset:1024 sc1" : "=v"(ar[S][1]) : "v"(pa) : "memory");
        asm volatile("global_load_dwordx4 %0, %1, off offset:2048 sc1" : "=v"(ar[S][2]) : "v"(pa) : "memory");
        asm volatile("global_load_dwordx4 %0, %1, off offset:3072 sc1" : "=v"(ar[S][3]) : "v"(pa) : "memory");
        lds_char* sb = smem + S * STAGE;
        const char* pb = gb + (size_t)t * (NF * FR_BYTES);
        lds_char* lb = sb + wid * 2048;
        __builtin_amdgcn_global_load_lds((g_cptr)pb, (l_ptr)lb, 16, 0, DMA_AUX);
        __builtin_amdgcn_global_load_lds((g_cptr)pb, (l_ptr)lb, 16, 1024, DMA_AUX);
        if constexpr (NFR >= 9) {
            const char* px = gx + (size_t)t * (NF * FR_BYTES);
            __builtin_amdgcn_global_load_lds((g_cptr)px, (l_ptr)(sb + 16384 + wid * 256), 4, 0, DMA_AUX);
            if constexpr (NFR >= 10)
                __builtin_amdgcn_global_load_lds((g_cptr)px, (l_ptr)(sb + 16384 + wid * 256), 4, 2048, DMA_AUX);
        }
    };
    auto rd = [&](h8 (&b)[NFR], int st, int kh) {
        lds_char* pb = smem + st * STAGE + (boff ^ (kh * 64));
        #pragma unroll
        for (int j = 0; j < NFR; ++j) b[j] = *(lds_h8*)(pb + j * 2048);
    };
    auto mma = [&](h8 (&a)[4], int kh, h8 (&b)[NFR]) {
        #pragma unroll
        for (int j = 0; j < NFR; ++j)
            #pragma unroll
            for (int i = 0; i < 2; ++i)
                acc[i][j] = __builtin_amdgcn_mfma_f32_16x16x32_f16(b[j], a[i * 2 + kh], acc[i][j], 0, 0, 0);
    };
    auto spin_flags = [&](int s) {
        const char* p = fl + (size_t)s * (NSWG * 4);
        for (unsigned it = 0;; ++it) {
            unsigned v;
            asm volatile("global_load_dword %0, %1, off sc0 sc1\n\ts_waitcnt vmcnt(0)" : "=v"(v) : "v"(p) : "memory");
            if (__all(v >= call)) break;
            if ((it & 15) == 15) {
                if (__hip_atomic_load(tmo, __ATOMIC_RELAXED, __HIP_MEMORY_SCOPE_AGENT) == call ||
                    __builtin_amdgcn_s_memrealtime() - t0 > DEADLINE) {
                    __hip_atomic_store(tmo, call, __ATOMIC_RELAXED, __HIP_MEMORY_SCOPE_AGENT);
                    break;
                }
            }
        }
    };
    auto pref_flags = [&](int s) {
        __builtin_amdgcn_global_load_lds((g_cptr)(fl + (size_t)s * (NSWG * 4)), (l_ptr)(smem + FLG_OFF), 4, 0, 17);
    };
    auto lds_flags_ok = [&]() {
        unsigned v;
        asm volatile("ds_read_b32 %0, %1\n\ts_waitcnt lgkmcnt(0)" : "=v"(v) : "v"((unsigned)(FLG_OFF + lane * 4)) : "memory");
        return __all(v >= call);
    };
    auto step = [&](int t, auto sc, auto arr_c, auto vm_c, auto iss_c) {
        constexpr int S = decltype(sc)::value, S1 = (S + 1) & 3, S3 = (S + 3) & 3;
        constexpr bool ARR = decltype(arr_c)::value;
        constexpr int VM = decltype(vm_c)::value;
        constexpr bool ISS = decltype(iss_c)::value;
        rd(b1, S, 1);
        mma(ar[S], 0, b0);
        #pragma unroll
        for (int g = 0; g < NFR; ++g) {
            __builtin_amdgcn_sched_group_barrier(0x008, 1, 0);
            __builtin_amdgcn_sched_group_barrier(0x100, 1, 0);
        }
        __builtin_amdgcn_sched_group_barrier(0x008, NFR, 0);
        __builtin_amdgcn_sched_barrier(0);
        if constexpr (ARR) {
            asm volatile("s_waitcnt vmcnt(%4) lgkmcnt(0)"
                         : "+v"(ar[S1][0]), "+v"(ar[S1][1]), "+v"(ar[S1][2]), "+v"(ar[S1][3]) : "i"(VM) : "memory");
            if constexpr (ISS) {
                if (remote && wid == 0) { if (!lds_flags_ok()) spin_flags(t + 3); }
            }
            BAR();
            __builtin_amdgcn_sched_barrier(0);
            if constexpr (ISS) {
                if (remote && wid == 0 && t + 4 < NKS) pref_flags(t + 4);
                issue(t + 3, std::integral_constant<int, S3>{});
            }
            rd(b0, S1, 0);
        }
        mma(ar[S], 1, b1);
        if constexpr (ARR) {
            #pragma unroll
            for (int g = 0; g < NFR; ++g) {
                __builtin_amdgcn_sched_group_barrier(0x008, 1, 1);
                if constexpr (ISS) { if (g < NBD) __builtin_amdgcn_sched_group_barrier(0x020, 1, 1); }
                __builtin_amdgcn_sched_group_barrier(0x100, 1, 1);
            }
            __builtin_amdgcn_sched_group_barrier(0x008, NFR, 1);
        }
        __builtin_amdgcn_sched_barrier(0);
    };
    using T_ = std::integral_constant<bool, true>;
    using F_ = std::integral_constant<bool, false>;
    using IP = std::integral_constant<int, PER>;
    using I0 = std::integral_constant<int, 0>;
    using S0 = std::integral_constant<int, 0>; using S1_ = std::integral_constant<int, 1>;
    using S2 = std::integral_constant<int, 2>; using S3_ = std::integral_constant<int, 3>;

    BAR();
    if (wid == 0) STAMP(2);
    if (remote && wid == 0) pref_flags(3);
    issue(0, S0{});
    issue(1, S1_{});
    issue(2, S2{});
    asm volatile("s_waitcnt vmcnt(%4)" : "+v"(ar[0][0]), "+v"(ar[0][1]), "+v"(ar[0][2]), "+v"(ar[0][3]) : "i"(2 * PER) : "memory");
    BAR();
    __builtin_amdgcn_sched_barrier(0);
    rd(b0, 0, 0);
    for (int t = 0; t < NKS - 4; t += 4) {
        step(t, S0{}, T_{}, IP{}, T_{});
        step(t + 1, S1_{}, T_{}, IP{}, T_{});
        step(t + 2, S2{}, T_{}, IP{}, T_{});
        step(t + 3, S3_{}, T_{}, IP{}, T_{});
    }
    step(NKS - 4, S0{}, T_{}, IP{}, T_{});
    step(NKS - 3, S1_{}, T_{}, IP{}, F_{});
    step(NKS - 2, S2{}, T_{}, I0{}, F_{});
    step(NKS - 1, S3_{}, F_{}, I0{}, F_{});
    if (wid == 0) STAMP(3);

    bad = bad || __hip_atomic_load(tmo, __ATOMIC_RELAXED, __HIP_MEMORY_SCOPE_AGENT) == call;
    const float qnan = __builtin_nanf("");
    #pragma unroll
    for (int i = 0; i < 2; ++i) {
        const int m = mt * BM + wid * 32 + i * 16 + (lane & 15);
        #pragma unroll
        for (int j = 0; j < NFR; ++j) {
            const int n = (f0 + j) * 16 + (lane >> 4) * 4;
            f4 o;
            #pragma unroll
            for (int r = 0; r < 4; ++r) o[r] = bad ? qnan : gelu_tanh(acc[i][j][r]);
            asm volatile("global_store_dwordx4 %0, %1, off sc1\n\ts_nop 1" :: "v"(out + (size_t)m * N + n), "v"(o) : "memory");
        }
    }
}

__global__ void __launch_bounds__(512, 2)
ag_gemm_kernel(const float* __restrict__ x, const float* __restrict__ w, float* __restrict__ out,
               void* const* __restrict__ ws_all, int rank)
{
    __shared__ __attribute__((aligned(16))) char smem_raw[LDS_BYTES];
    lds_char* smem = (lds_char*)smem_raw;

    const int tid = threadIdx.x, lane = tid & 63;
    const int wid = __builtin_amdgcn_readfirstlane(tid >> 6);
    const int bid = blockIdx.x;
    const int xcd = bid & 7, idx = bid >> 3;
    char* ws = (char*)ws_all[rank];
    const unsigned call = __builtin_amdgcn_readfirstlane(
        __hip_atomic_load((g_u32*)(ws + OFF_EPOCH), __ATOMIC_RELAXED, __HIP_MEMORY_SCOPE_AGENT)) + 1;
    const size_t x16_off = OFF_X16 + (size_t)(call & 1) * X16_BYTES;
    if (wid == 0) STAMP(0);

    if (idx < GPX) {
        const int gwv = (xcd * GPX + idx) * 8 + wid;
        constexpr int NWV = NGW * 8;
        for (int i = 0; i < GDELAY; ++i) __builtin_amdgcn_s_sleep(32);
        {
            const int kg = lane >> 3, ng = lane & 7;
            f4 v[3][8], u[3][2];
            #pragma unroll
            for (int it = 0; it < 3; ++it) {
                const int b = gwv + it * NWV;
                if (b < 4096) {
                    const int ks = b >> 6, nb = b & 63;
                    const float* p = w + (size_t)(ks * 64 + kg * 8) * N + nb * 32 + ng * 4;
                    #pragma unroll
                    for (int j = 0; j < 8; ++j) v[it][j] = __builtin_nontemporal_load((const f4*)(p + (size_t)j * N));
                    const int xkh = b & 127, rf = b >> 7;
                    const float* px = x + (size_t)(rf * 16 + (lane & 15)) * K + xkh * 32 + (lane >> 4) * 8;
                    u[it][0] = __builtin_nontemporal_load((const f4*)px); u[it][1] = __builtin_nontemporal_load((const f4*)(px + 4));
                }
            }
            #pragma unroll
            for (int it = 0; it < 3; ++it) {
                const int b = gwv + it * NWV;
                if (b < 4096) {
                    const int ks = b >> 6, nb = b & 63;
                    #pragma unroll
                    for (int i = 0; i < 4; ++i) {
                        h8 h;
                        #pragma unroll
                        for (int j = 0; j < 8; ++j) h[j] = (_Float16)v[it][j][i];
                        const int n = nb * 32 + ng * 4 + i;
                        char* dst = ws + OFF_W16 + (size_t)(ks * NF + (n >> 4)) * FR_BYTES + (n & 15) * 128 + ((kg ^ (n & 7)) * 16);
                        asm volatile("global_store_dwordx4 %0, %1, off sc1\n\ts_nop 1" :: "v"(dst), "v"(h) : "memory");
                    }
                    const int xkh = b & 127, rf = b >> 7;
                    h8 h;
                    #pragma unroll
                    for (int e = 0; e < 4; ++e) { h[e] = (_Float16)u[it][0][e]; h[4 + e] = (_Float16)u[it][1][e]; }
                    char* dst = ws + x16_off + (size_t)((rank * 2 + (rf >> 4)) * NKS + (xkh >> 1)) * A_BYTES
                              + ((rf & 15) * 2 + (xkh & 1)) * 1024 + lane * 16;
                    asm volatile("global_store_dwordx4 %0, %1, off sc1\n\ts_nop 1" :: "v"(dst), "v"(h) : "memory");
                }
            }
            asm volatile("s_waitcnt vmcnt(0)" ::: "memory");
            BAR();
            if (wid == 0) STAMP(1);
        }
        bool bad = false;
        const int mt = xcd * 2 + idx / GPR, g = idx % GPR;
        if (wid == 0) {
            if (lane == 0)
                __hip_atomic_fetch_add((g_u32*)(ws + OFF_LCNT + xcd * 128), 1u, __ATOMIC_RELAXED, __HIP_MEMORY_SCOPE_AGENT);
            constexpr int FPT = BM / SROWS;
            const int src = mt >> 1;
            const char* pa = ws + OFF_LCNT + (lane & 7) * 128;
            unsigned thr = lane < 8 ? call * GPX : 0u;
            if (src != rank && lane >= 8 && lane < 8 + 3 * FPT) {
                const int k = (lane - 8) / FPT, j = (lane - 8) % FPT;
                pa = ws + OFF_XFLAG + (size_t)src * XFLAG_SRC + (size_t)(k * NSWG + (mt & 1) * FPT + j) * 4;
                thr = call;
            }
            g_u32* tmo = (g_u32*)(ws + OFF_TMO);
            const unsigned long long t0 = __builtin_amdgcn_s_memrealtime();
            for (unsigned it = 0;; ++it) {
                unsigned v;
                asm volatile("global_load_dword %0, %1, off sc0 sc1\n\ts_waitcnt vmcnt(0)" : "=v"(v) : "v"(pa) : "memory");
                if (__all(v >= thr)) break;
                if ((it & 15) == 15) {
                    if (__hip_atomic_load(tmo, __ATOMIC_RELAXED, __HIP_MEMORY_SCOPE_AGENT) == call ||
                        __builtin_amdgcn_s_memrealtime() - t0 > DEADLINE) {
                        __hip_atomic_store(tmo, call, __ATOMIC_RELAXED, __HIP_MEMORY_SCOPE_AGENT);
                        break;
                    }
                }
            }
        }
        if (g < NWIDE) gemm_tile<FLO + 1>(smem, ws, out, mt, (FLO + 1) * g, wid, lane, call, rank, bid, bad);
        else           gemm_tile<FLO>(smem, ws, out, mt, (FLO + 1) * NWIDE + FLO * (g - NWIDE), wid, lane, call, rank, bid, bad);
    } else {
        const int swg = xcd * SPX + (idx - GPX);
        const int r0 = swg * SROWS;
        const unsigned A_LOADED = CTL_OFF, A_FREED = CTL_OFF + 64;
        if (tid < 16) *(lds_u32*)(smem + CTL_OFF + tid * 4 + (tid ? 60 : 0)) = 0;
        __syncthreads();
        if (wid == 7) {
            const char* gsrc = (const char*)x + (size_t)r0 * (K * 4) + lane * 16;
            for (int c = 0; c < NCH; ++c) {
                if (c >= SRING) {
                    for (;;) {
                        unsigned v;
                        asm volatile("ds_read_b32 %0, %1\n\ts_waitcnt lgkmcnt(0)" : "=v"(v) : "v"(A_FREED + (c % SRING) * 4) : "memory");
                        if (__builtin_amdgcn_readfirstlane(v) >= (unsigned)((c / SRING) * SPC)) break;
                        __builtin_amdgcn_s_sleep(1);
                    }
                }
                lds_char* slot = smem + (c % SRING) * (SROWS * SROW_LDS);
                #pragma unroll
                for (int r = 0; r < SROWS; ++r)
                    __builtin_amdgcn_global_load_lds((g_cptr)(gsrc + (size_t)r * (K * 4) + c * 1024), (l_ptr)(slot + r * SROW_LDS), 16, 0, 2);
                if (c > 0) {
                    asm volatile("s_waitcnt vmcnt(%0)" :: "i"(SROWS) : "memory");
                    asm volatile("ds_write_b32 %0, %1" :: "v"(A_LOADED), "v"((unsigned)c) : "memory");
                }
            }
            asm volatile("s_waitcnt vmcnt(0)" ::: "memory");
            asm volatile("ds_write_b32 %0, %1" :: "v"(A_LOADED), "v"((unsigned)NCH) : "memory");
        } else if (wid < NSW) {
            const char* d0 = (const char*)ws_all[(rank + 1) & 7];
            const char* d1 = (const char*)ws_all[(rank + 2) & 7];
            const char* d2 = (const char*)ws_all[(rank + 3) & 7];
            const char* d3 = (const char*)ws_all[(rank + 4) & 7];
            const char* d4 = (const char*)ws_all[(rank + 5) & 7];
            const char* d5 = (const char*)ws_all[(rank + 6) & 7];
            const char* d6 = (const char*)ws_all[(rank + 7) & 7];
            char* fdst = (char*)ws_all[(rank + 1 + (lane & 7)) & 7] + OFF_XFLAG + (size_t)rank * XFLAG_SRC + (wid * NSWG + swg) * 4;
            asm volatile("" : "+v"(fdst));
            const unsigned lrd = (lane & 15) * SROW_LDS + (lane >> 4) * 32;
            const int mtg = rank * 2 + (r0 >> 8);
            unsigned doff = (unsigned)(x16_off + ((size_t)mtg * NKS + wid) * A_BYTES + ((r0 & 255) >> 4) * 2048 + lane * 16);
            constexpr int NRG = SROWS / 8;
            #define ST1(d) asm volatile("global_store_dwordx4 %0, %1, %2 offset:%3 sc0 sc1\n\ts_nop 1" :: "v"(doff), "v"(h), "s"(d), "i"(rg * 1024) : "memory")
            for (int ks = wid; ks < NKS; ks += NSW) {
                const int c = ks / SPC;
                for (;;) {
                    unsigned v;
                    asm volatile("ds_read_b32 %0, %1\n\ts_waitcnt lgkmcnt(0)" : "=v"(v) : "v"(A_LOADED) : "memory");
                    if (__builtin_amdgcn_readfirstlane(v) > (unsigned)c) break;
                    __builtin_amdgcn_s_sleep(1);
                }
                const unsigned sb = (c % SRING) * (SROWS * SROW_LDS) + (ks % SPC) * 256 + lrd;
                f4 v[NRG][2];
                #pragma unroll
                for (int rg = 0; rg < NRG; ++rg) {
                    asm volatile("ds_read_b128 %0, %1 offset:%2" : "=v"(v[rg][0]) : "v"(sb), "i"((rg >> 1) * 16 * SROW_LDS + (rg & 1) * 128) : "memory");
                    asm volatile("ds_read_b128 %0, %1 offset:%2" : "=v"(v[rg][1]) : "v"(sb), "i"((rg >> 1) * 16 * SROW_LDS + (rg & 1) * 128 + 16) : "memory");
                }
                #pragma unroll
                for (int rg = 0; rg < NRG; ++rg) asm volatile("s_waitcnt lgkmcnt(0)" : "+v"(v[rg][0]), "+v"(v[rg][1]) :: "memory");
                if (lane == 0) asm volatile("ds_add_u32 %0, %1" :: "v"(A_FREED + (c % SRING) * 4), "v"(1u) : "memory");
                if (ks == 0) LSTAMP(16);
                #pragma unroll
                for (int rg = 0; rg < NRG; ++rg) {
                    h8 h;
                    #pragma unroll
                    for (int e = 0; e < 4; ++e) { h[e] = (_Float16)v[rg][0][e]; h[4 + e] = (_Float16)v[rg][1][e]; }
                    ST1(d0); ST1(d1); ST1(d2); ST1(d3); ST1(d4); ST1(d5); ST1(d6);
                }
                doff += NSW * A_BYTES;
                asm volatile("s_waitcnt vmcnt(0)" ::: "memory");
                if (ks == 0) LSTAMP(17);
                if (ks == 1) LSTAMP(18);
                if (ks == 2) LSTAMP(19);
                if (lane < 7) asm volatile("global_store_dword %0, %1, off sc0 sc1" :: "v"(fdst), "v"(call) : "memory");
                fdst += NSW * NSWG * 4;
                if ((ks % SPC) == SPC - 1) LSTAMP(c);
            }
            #undef ST1
        }
    }
    if (wid == 0 && lane == 0) {
        unsigned old = __hip_atomic_fetch_add((g_u32*)(ws + OFF_DONE), 1u, __ATOMIC_RELAXED, __HIP_MEMORY_SCOPE_AGENT);
        if ((old + 1) % NWG == 0)
            __hip_atomic_store((g_u32*)(ws + OFF_EPOCH), (old + 1) / NWG, __ATOMIC_RELAXED, __HIP_MEMORY_SCOPE_AGENT);
    }
}

extern "C" void kernel_launch(int rank, int nranks, ncclComm_t comm,
                              hipStream_t stream,
                              void* const* d_in, const int* in_sizes, int n_in,
                              void* d_out, int out_size,
                              const int* in_dtypes, int out_dtype,
                              void* const* d_out_all,
                              void* const* d_ws_all, size_t ws_bytes) {
    if (nranks != NRANK || n_in < 2 || ws_bytes < WS_NEED || in_sizes[0] != MS * K ||
        in_sizes[1] != K * N || out_size != M * N) {
        fprintf(stderr, "kernel_launch: unexpected problem geometry\n");
        return;
    }
    hipLaunchKernelGGL(ag_gemm_kernel, dim3(NWG), dim3(512), 0, stream,
                       (const float*)d_in[0], (const float*)d_in[1], (float*)d_out,
                       d_ws_all, rank);
    HIP_CHECK(hipGetLastError());
}
```
